```python
import numpy as np

kernel_name = "gpu_gemm_rs_m4096_k4096_n32768_bf16_gelu"


ABSMAX_RTOL = 0.008

COMPUTE_DTYPE = "bfloat16"
EPILOGUE = "gelu"
IN_QUANT = None
OUT_QUANT = None
IN_DTYPES = {"x": "bfloat16", "w_mat": "bfloat16"}
WORLD_SIZE = 8

def _to_bf16(x):
    u = np.ascontiguousarray(x, dtype=np.float32).view(np.uint32)
    return ((u + 0x7FFF + ((u >> 16) & 1)) & 0xFFFF0000).view(np.float32)

def setup_inputs():

    x = _to_bf16(np.random.randn(4096, 4096).astype(np.float32))
    w_mat = _to_bf16((np.random.randn(4096, 32768) / np.sqrt(4096)).astype(np.float32))
    return {"x": x, "w_mat": w_mat}

def _epilogue(y):
    c = 0.7978845608028654
    return (0.5 * y * (1.0 + np.tanh(c * (y + 0.044715 * y ** 3)))).astype(np.float32)

def reference(x, w_mat):
    y = (x @ w_mat).astype(np.float32)
    return _epilogue(y)
```

```cpp
#include <hip/hip_runtime.h>
#include <rccl/rccl.h>
#include <cstdio>
#include <cstdint>

#define HIP_CHECK(expr) do { hipError_t _e = (expr); if (_e != hipSuccess) { \
    fprintf(stderr, "HIP error %s at %s:%d\n", hipGetErrorString(_e), __FILE__, __LINE__); \
    return; } } while (0)

namespace {
#define LAS __attribute__((address_space(3)))
typedef unsigned short bf16_t;
typedef short bf16x8 __attribute__((ext_vector_type(8)));
typedef float f32x4 __attribute__((ext_vector_type(4)));
typedef unsigned u32x4 __attribute__((ext_vector_type(4)));

constexpr int NR = 8;
constexpr int KS = 512;
constexpr int NTOT = 32768;
constexpr int NSL = 4096;
constexpr int NKT = 64;
constexpr int HTB = 16384;
constexpr int STAGE_BYTES = 8 * HTB;
constexpr int SCR_BYTES = 32768;
constexpr int LDS_BYTES = STAGE_BYTES + SCR_BYTES;
constexpr size_t CTL_BYTES = 1u << 20;
constexpr size_t OPER_BYTES = 32u << 20;
constexpr size_t BUF_BYTES = 2 * OPER_BYTES;
constexpr size_t KT_BYTES = 32 * (size_t)HTB;
constexpr size_t WS_NEED = CTL_BYTES + 2 * BUF_BYTES;
constexpr int W_TICKET = 0, W_TMO = 32, W_CNT = 64  , W_OUT = 64 + 32 * 8, W_DUTY = 512  ;
constexpr unsigned long long DEADLINE = 100000000ull;

struct Params {
    const bf16_t* x; const bf16_t* w;
    char* ws[NR]; char* out[NR];
    int rank;
};

__device__ __forceinline__ void st_sys(void* p, u32x4 v) {
    asm volatile("global_store_dwordx4 %0, %1, off sc0 sc1\n\ts_nop 1" :: "v"(p), "v"(v) : "memory");
}
__device__ __forceinline__ unsigned cvt_pk_bf16(float lo, float hi) { unsigned r; asm volatile("v_cvt_pk_bf16_f32 %0, %1, %2" : "=v"(r) : "v"(lo), "v"(hi)); return r; }
__device__ __forceinline__ float gelu1(float y) {
    const float k1 = -2.302208198144325f;
    const float k3 = k1 * 0.044715f;
    const float t = y * (k1 + k3 * y * y);
    const float e = __builtin_amdgcn_exp2f(t);
    return y * __builtin_amdgcn_rcpf(1.0f + e);
}

__device__ __forceinline__ void wait_ge(const unsigned* p, unsigned need, unsigned* tmo, unsigned call) {
    const unsigned long long t0 = __builtin_amdgcn_s_memrealtime();
    while ((unsigned)__builtin_amdgcn_readfirstlane(__hip_atomic_load(p, __ATOMIC_RELAXED, __HIP_MEMORY_SCOPE_SYSTEM)) < need) {
        if ((unsigned)__builtin_amdgcn_readfirstlane(__hip_atomic_load(tmo, __ATOMIC_RELAXED, __HIP_MEMORY_SCOPE_AGENT)) == call
            || __builtin_amdgcn_s_memrealtime() - t0 > DEADLINE) {
            __hip_atomic_store(tmo, call, __ATOMIC_RELAXED, __HIP_MEMORY_SCOPE_AGENT);
            break;
        }
        __builtin_amdgcn_s_sleep(2);
    }
}

__device__ __forceinline__ void push_block(const Params& P, int s, int d, int blk, size_t bufoff, LAS unsigned char* scr, int lane) {
    char* wsd = P.ws[d];
    const int kt = s * 8 + P.rank;
    {
        const bf16_t* xs = P.x + (size_t)(blk * 128 + (lane & 15)) * KS + s * 64 + (lane >> 4) * 8;
        char* dst = wsd + bufoff + (size_t)kt * KT_BYTES + (size_t)blk * HTB + lane * 16;
#pragma unroll
        for (int h = 0; h < 2; ++h) {
            u32x4 v[8];
#pragma unroll
            for (int i = 0; i < 8; ++i) { const int f = h * 8 + i; v[i] = *(const u32x4*)(xs + (size_t)(f >> 1) * 16 * KS + (f & 1) * 32); }
#pragma unroll
            for (int i = 0; i < 8; ++i) st_sys(dst + (h * 8 + i) * 1024, v[i]);
        }
    }
    {
        const bf16_t* wsrc = P.w + (size_t)(s * 64 + (lane >> 3)) * NTOT + d * NSL + blk * 128 + (lane & 7) * 8;
        char* dst = wsd + bufoff + OPER_BYTES + (size_t)kt * KT_BYTES + (size_t)blk * HTB + lane * 16;
        const int i16 = lane & 15, g = lane >> 4;
#pragma unroll
        for (int piece = 0; piece < 4; ++piece) {
            const int kb = piece >> 1, nh = piece & 1;
            u32x4 v[4];
#pragma unroll
            for (int i = 0; i < 4; ++i) v[i] = *(const u32x4*)(wsrc + (size_t)(kb * 32 + i * 8) * NTOT + nh * 64);
#pragma unroll
            for (int i = 0; i < 4; ++i) {
                const int r = (lane >> 3) + 8 * i;
                *(LAS u32x4*)(scr + r * 128 + (((lane & 7) ^ ((i & 1) << 2)) * 16)) = v[i];
            }
#pragma unroll
            for (int q = 0; q < 4; ++q) {
                const int cl = (q >> 1) * 32 + 8 * (i16 >> 2) + 4 * (q & 1) + (i16 & 3);
                const LAS unsigned short* src = (const LAS unsigned short*)(scr + (g * 8) * 128 + (((cl >> 3) ^ ((g & 1) << 2)) * 16) + (cl & 7) * 2);
                u32x4 o;
                o.x = (unsigned)src[0 * 64] | ((unsigned)src[1 * 64] << 16);
                o.y = (unsigned)src[2 * 64] | ((unsigned)src[3 * 64] << 16);
                o.z = (unsigned)src[4 * 64] | ((unsigned)src[5 * 64] << 16);
                o.w = (unsigned)src[6 * 64] | ((unsigned)src[7 * 64] << 16);
                st_sys(dst + ((nh * 4 + q) * 2 + kb) * 1024, o);
            }
        }
    }
}

__global__ __launch_bounds__(512, 2) void fused_gemm(Params P) {
    extern __shared__ __attribute__((aligned(16))) unsigned char shm[];
    LAS unsigned char* lds = (LAS unsigned char*)shm;
    const int tid = threadIdx.x, wid = __builtin_amdgcn_readfirstlane(tid >> 6), lane = tid & 63;
    const int wr = wid >> 2, wc = wid & 3, fr = lane & 15, fq = lane >> 4;
    const int b = blockIdx.x, xcd = b & 7, idx = b >> 3;
    const int rank = P.rank;
    unsigned* ctl = (unsigned*)P.ws[rank];
    unsigned* tmo = ctl + W_TMO;
    if (tid == 0) {
        const unsigned t = __hip_atomic_fetch_add(ctl + W_TICKET, 1u, __ATOMIC_RELAXED, __HIP_MEMORY_SCOPE_AGENT);
        *(LAS unsigned*)(lds + STAGE_BYTES) = t / gridDim.x + 1u;
    }
    __syncthreads();
    const unsigned call = (unsigned)__builtin_amdgcn_readfirstlane(*(LAS unsigned*)(lds + STAGE_BYTES));
    __syncthreads();
    const size_t bufoff = CTL_BYTES + (size_t)(call & 1u) * BUF_BYTES;
    const int dest = (rank + xcd) & 7;
    const int pm = dest * 2 + (idx & 1), pn = idx >> 1;
    const int duty_s = idx >> 2, duty_blk = (idx & 3) * 8 + wid;
    LAS unsigned char* scr = lds + STAGE_BYTES + wid * 4096;

    auto hook = [&](int h) {
        const int duty_h = duty_s == 0 ? 0 : duty_s - 1;
        if (duty_h == h) {
            if (duty_s > 0) wait_ge(ctl + W_DUTY + 32 * ((duty_s - 1) * 8 + xcd), call * 32u, tmo, call);
            push_block(P, duty_s, dest, duty_blk, bufoff, scr, lane);
            asm volatile("s_waitcnt vmcnt(0)" ::: "memory");
            __builtin_amdgcn_fence(__ATOMIC_RELEASE, "");
            asm volatile("s_waitcnt vmcnt(0)" ::: "memory");
            if (lane == 0) {
                __hip_atomic_fetch_add((unsigned*)P.ws[dest] + W_CNT + 32 * duty_s, 1u, __ATOMIC_RELAXED, __HIP_MEMORY_SCOPE_SYSTEM);
                __hip_atomic_fetch_add(ctl + W_DUTY + 32 * (duty_s * 8 + xcd), 1u, __ATOMIC_RELAXED, __HIP_MEMORY_SCOPE_AGENT);
            }
        }
        if (wid == 0) {
            wait_ge(ctl + W_CNT + 32 * h, call * 256u, tmo, call);
            __builtin_amdgcn_fence(__ATOMIC_ACQUIRE, "");
            asm volatile("s_waitcnt vmcnt(0)" ::: "memory");
        }
        asm volatile("" ::: "memory"); __builtin_amdgcn_s_barrier(); asm volatile("" ::: "memory");
    };

    const unsigned voff0 = (unsigned)tid * 16u, voff1 = voff0 + 8192u;
    const unsigned ldsw = (unsigned)wid * 1024u;
    const int aoff = wr * 8192 + lane * 16, boff = wc * 4096 + lane * 16;
#define SA(b_, h_) (((b_) * 2 + (h_)) * HTB)
#define SB(b_, h_) ((4 + (b_) * 2 + (h_)) * HTB)
#define STAGE(bufo, gbase) do { \
        __builtin_amdgcn_global_load_lds((const unsigned*)((const char*)(gbase) + voff0), (LAS unsigned*)(lds + (bufo) + ldsw), 16, 0, 0); \
        __builtin_amdgcn_global_load_lds((const unsigned*)((const char*)(gbase) + voff1), (LAS unsigned*)(lds + (bufo) + ldsw + 8192), 16, 0, 0); } while (0)
#define LDA(dst, b_, h_) do { _Pragma("unroll") for (int m = 0; m < 4; ++m) _Pragma("unroll") for (int k = 0; k < 2; ++k) dst[m][k] = *(const LAS bf16x8*)(lds + SA(b_, h_) + aoff + m * 2048 + k * 1024); } while (0)
#define LDB(dst, b_, h_) do { _Pragma("unroll") for (int n = 0; n < 2; ++n) _Pragma("unroll") for (int k = 0; k < 2; ++k) dst[n][k] = *(const LAS bf16x8*)(lds + SB(b_, h_) + boff + n * 2048 + k * 1024); } while (0)
#define MMA(ai, bj, At_, Bt_) do { __builtin_amdgcn_s_setprio(1); _Pragma("unroll") for (int m = 0; m < 4; ++m) _Pragma("unroll") for (int n = 0; n < 2; ++n) _Pragma("unroll") for (int k = 0; k < 2; ++k) \
        acc[ai][bj][m][n] = __builtin_amdgcn_mfma_f32_16x16x32_bf16(Bt_[n][k], At_[m][k], acc[ai][bj][m][n], 0, 0, 0); __builtin_amdgcn_s_setprio(0); } while (0)
#define WAIT_V(n) asm volatile("s_waitcnt vmcnt(" #n ")" ::: "memory")
#define WAIT_L(n) asm volatile("s_waitcnt lgkmcnt(" #n ")" ::: "memory")
#define BAR __builtin_amdgcn_s_barrier()
#define SCHED __builtin_amdgcn_sched_barrier(0)

    f32x4 acc[2][2][4][2];
#pragma unroll
    for (int a = 0; a < 2; ++a)
#pragma unroll
        for (int c = 0; c < 2; ++c)
#pragma unroll
            for (int m = 0; m < 4; ++m)
#pragma unroll
                for (int n = 0; n < 2; ++n) acc[a][c][m][n] = (f32x4){0.f, 0.f, 0.f, 0.f};
    bf16x8 At[4][2], B0[2][2], B1[2][2];
    const char* cA = P.ws[rank] + bufoff + (size_t)pm * 2 * HTB;
    const char* cB = P.ws[rank] + bufoff + OPER_BYTES + (size_t)pn * 2 * HTB;
    constexpr size_t kstep = KT_BYTES, hstep = HTB;

    hook(0);
    STAGE(SB(0, 0), cB); STAGE(SA(0, 0), cA); STAGE(SB(0, 1), cB + hstep); STAGE(SA(0, 1), cA + hstep);
    if (wr == 1) BAR;
    WAIT_V(4); BAR;
    STAGE(SB(1, 0), cB + kstep); STAGE(SA(1, 0), cA + kstep); STAGE(SB(1, 1), cB + hstep + kstep);
    WAIT_V(6); BAR;
    for (int t = 0; t < NKT; t += 2) {
        const bool last = (t == NKT - 2);
        const char* a1 = cA + (size_t)(t + 1) * kstep;
        const char* a2 = last ? cA : cA + (size_t)(t + 2) * kstep; const char* b2 = last ? cB : cB + (size_t)(t + 2) * kstep;
        const char* a3 = a2 + kstep; const char* b3 = b2 + kstep;
        if (((t + 2) & 7) == 0 && !last) hook((t + 2) >> 3);
        LDB(B0, 0, 0); SCHED; LDA(At, 0, 0); STAGE(SA(1, 1), a1 + hstep);
        WAIT_L(8); BAR; WAIT_L(0); MMA(0, 0, At, B0); BAR; SCHED;
        LDB(B1, 0, 1); STAGE(SB(0, 0), b2);
        BAR; WAIT_L(0); MMA(0, 1, At, B1); BAR;
        LDA(At, 0, 1); STAGE(SA(0, 0), a2);
        BAR; WAIT_L(0); MMA(1, 0, At, B0); BAR; SCHED;
        STAGE(SB(0, 1), b2 + hstep);
        WAIT_V(6); BAR; MMA(1, 1, At, B1); BAR;
        LDB(B0, 1, 0); SCHED; LDA(At, 1, 0); STAGE(SA(0, 1), a2 + hstep);
        WAIT_L(8); BAR; WAIT_L(0); MMA(0, 0, At, B0); BAR; SCHED;
        LDB(B1, 1, 1); STAGE(SB(1, 0), b3);
        BAR; WAIT_L(0); MMA(0, 1, At, B1); BAR;
        LDA(At, 1, 1); STAGE(SA(1, 0), a3);
        BAR; WAIT_L(0); MMA(1, 0, At, B0); BAR; SCHED;
        STAGE(SB(1, 1), b3 + hstep);
        WAIT_V(6); BAR; MMA(1, 1, At, B1); BAR;
    }
    WAIT_V(0);
    if (wr == 0) BAR;
    BAR;
    {
        const bool bad = (unsigned)__builtin_amdgcn_readfirstlane(__hip_atomic_load(tmo, __ATOMIC_RELAXED, __HIP_MEMORY_SCOPE_AGENT)) == call;
        char* obase = P.out[dest] + ((size_t)((pm & 1) * 256 + wr * 64 + fr) * NTOT + (size_t)rank * NSL + pn * 256 + wc * 32 + 8 * fq) * 2;
#pragma unroll
        for (int ai = 0; ai < 2; ++ai)
#pragma unroll
            for (int m = 0; m < 4; ++m) {
                char* rowp = obase + (size_t)(ai * 128 + m * 16) * NTOT * 2;
#pragma unroll
                for (int bj = 0; bj < 2; ++bj) {
                    const f32x4 v0 = acc[ai][bj][m][0], v1 = acc[ai][bj][m][1];
                    u32x4 o;
                    o.x = cvt_pk_bf16(gelu1(v0[0]), gelu1(v0[1])); o.y = cvt_pk_bf16(gelu1(v0[2]), gelu1(v0[3]));
                    o.z = cvt_pk_bf16(gelu1(v1[0]), gelu1(v1[1])); o.w = cvt_pk_bf16(gelu1(v1[2]), gelu1(v1[3]));
                    if (bad) o = (u32x4){0x7fc07fc0u, 0x7fc07fc0u, 0x7fc07fc0u, 0x7fc07fc0u};
                    st_sys(rowp + bj * 256, o);
                }
            }
    }
    asm volatile("s_waitcnt vmcnt(0)" ::: "memory");
    __syncthreads();
    if (tid == 0) {
        __builtin_amdgcn_fence(__ATOMIC_RELEASE, "");
        asm volatile("s_waitcnt vmcnt(0)" ::: "memory");
        __hip_atomic_fetch_add((unsigned*)P.ws[dest] + W_OUT, 1u, __ATOMIC_RELAXED, __HIP_MEMORY_SCOPE_SYSTEM);
    }
    if (b == 0 && wid == 0) {
        wait_ge(ctl + W_OUT, call * 256u, tmo, call);
        __builtin_amdgcn_fence(__ATOMIC_ACQUIRE, "");
        if ((unsigned)__builtin_amdgcn_readfirstlane(__hip_atomic_load(tmo, __ATOMIC_RELAXED, __HIP_MEMORY_SCOPE_AGENT)) == call && lane == 0)
            *(unsigned short*)P.out[rank] = (unsigned short)0x7fc0u;
    }
}
}

extern "C" void kernel_launch(int rank, int nranks, ncclComm_t comm,
                              hipStream_t stream,
                              void* const* d_in, const int* in_sizes, int n_in,
                              void* d_out, int out_size,
                              const int* in_dtypes, int out_dtype,
                              void* const* d_out_all,
                              void* const* d_ws_all, size_t ws_bytes) {
    if (nranks != NR || ws_bytes < WS_NEED || n_in < 2) { fprintf(stderr, "unexpected configuration\n"); return; }
    static bool attr = false;
    if (!attr) { HIP_CHECK(hipFuncSetAttribute((const void*)fused_gemm, hipFuncAttributeMaxDynamicSharedMemorySize, LDS_BYTES)); attr = true; }
    Params P;
    P.x = (const bf16_t*)d_in[0]; P.w = (const bf16_t*)d_in[1]; P.rank = rank;
    for (int p = 0; p < NR; ++p) { P.ws[p] = (char*)d_ws_all[p]; P.out[p] = (char*)d_out_all[p]; }
    hipLaunchKernelGGL(fused_gemm, dim3(256), dim3(512), LDS_BYTES, stream, P);
    HIP_CHECK(hipGetLastError());
}
```

```cpp
#include <hip/hip_runtime.h>
#include <rccl/rccl.h>
#include <cstdio>
#include <cstdint>

#define HIP_CHECK(expr) do { hipError_t _e = (expr); if (_e != hipSuccess) { \
    fprintf(stderr, "HIP error %s at %s:%d\n", hipGetErrorString(_e), __FILE__, __LINE__); \
    return; } } while (0)

namespace {
#define LAS __attribute__((address_space(3)))
typedef unsigned short bf16_t;
typedef short bf16x8 __attribute__((ext_vector_type(8)));
typedef float f32x4 __attribute__((ext_vector_type(4)));
typedef unsigned u32x4 __attribute__((ext_vector_type(4)));

#define SPLIT 3
#define NWG 4
#define HANDOFF 16
#define RELFENCE 0
constexpr int NR = 8;
constexpr int KS = 512;
constexpr int NTOT = 32768;
constexpr int NSL = 4096;
constexpr int NKT = 64;
constexpr int HTB = 16384;
constexpr int STAGE_BYTES = 8 * HTB;
constexpr int SCR_BYTES = 256;
constexpr int LDS_BYTES = STAGE_BYTES + SCR_BYTES;
constexpr size_t CTL_BYTES = 1u << 20;
constexpr size_t OPER_BYTES = 32u << 20;
constexpr size_t BUF_BYTES = 2 * OPER_BYTES;
constexpr size_t KT_BYTES = 32 * (size_t)HTB;
constexpr size_t WS_NEED = CTL_BYTES + 2 * BUF_BYTES;
constexpr int W_TICKET = 0, W_TMO = 32, W_OUT = 64, W_CNT = 128  , W_DUTY = 1024  ;
constexpr unsigned long long DEADLINE = 100000000ull;

#define STAMP(i) do { } while (0)

struct Params {
    const bf16_t* x; const bf16_t* w;
    char* ws[NR]; char* out[NR];
    int rank;
};

__device__ __forceinline__ void st_sys(void* p, u32x4 v) {
    asm volatile("global_store_dwordx4 %0, %1, off sc0 sc1\n\ts_nop 1" :: "v"(p), "v"(v) : "memory");
}
__device__ __forceinline__ unsigned cvt_pk_bf16(float lo, float hi) { unsigned r; asm volatile("v_cvt_pk_bf16_f32 %0, %1, %2" : "=v"(r) : "v"(lo), "v"(hi)); return r; }
__device__ __forceinline__ float gelu1(float y) {
    const float k1 = -2.302208198144325f;
    const float k3 = k1 * 0.044715f;
    const float t = y * (k1 + k3 * y * y);
    const float e = __builtin_amdgcn_exp2f(t);
    return y * __builtin_amdgcn_rcpf(1.0f + e);
}

__device__ __forceinline__ void wait_ge(const unsigned* p, unsigned need, unsigned* tmo, unsigned call) {
    const unsigned long long t0 = __builtin_amdgcn_s_memrealtime();
    while ((unsigned)__builtin_amdgcn_readfirstlane(__hip_atomic_load(p, __ATOMIC_RELAXED, __HIP_MEMORY_SCOPE_SYSTEM)) < need) {
        if ((unsigned)__builtin_amdgcn_readfirstlane(__hip_atomic_load(tmo, __ATOMIC_RELAXED, __HIP_MEMORY_SCOPE_AGENT)) == call
            || __builtin_amdgcn_s_memrealtime() - t0 > DEADLINE) {
            __hip_atomic_store(tmo, call, __ATOMIC_RELAXED, __HIP_MEMORY_SCOPE_AGENT);
            break;
        }
        __builtin_amdgcn_s_sleep(2);
    }
}

__device__ __forceinline__ void unit_load(const Params& P, int s, int d, int r, int lane, LAS unsigned char* slot) {
    const bool isA = r < 64;
    const int blk = (r - 64) >> 1, kb = r & 1;
    const char* base = isA ? (const char*)(P.x + (size_t)(r * 64) * KS + s * 64) : (const char*)(P.w + (size_t)(s * 64 + kb * 32) * NTOT + d * NSL + blk * 128);
    const unsigned l0 = isA ? (unsigned)((lane & 15) * KS + (lane >> 4) * 8) * 2u : (unsigned)((lane >> 3) * NTOT + (lane & 7) * 8) * 2u;
    const unsigned l1 = isA ? l0 : (unsigned)((lane >> 3) * NTOT + ((lane & 7) ^ 4) * 8) * 2u;
    const unsigned s1 = isA ? 16u * KS * 2u : 8u * NTOT * 2u, s2 = isA ? 64u : 128u;
    const unsigned d1 = isA ? 2048u : 1024u, d2 = isA ? 1024u : 4096u;
#pragma unroll
    for (int f = 0; f < 8; ++f)
        __builtin_amdgcn_global_load_lds((const unsigned*)(base + ((((f >> 1) & 1) ? l1 : l0) + (f >> 1) * s1 + (f & 1) * s2)),
                                         (LAS unsigned*)(slot + (f >> 1) * d1 + (f & 1) * d2), 16, 0, 0);
}
__device__ __forceinline__ void unit_store(char* dbase, int kt, int r, const LAS unsigned char* slot, int lane) {
    if (r < 64) {
        char* dst = dbase + (size_t)kt * KT_BYTES + (size_t)r * (HTB / 2) + lane * 16;
#pragma unroll
        for (int f = 0; f < 8; ++f) st_sys(dst + f * 1024, *(const LAS u32x4*)(slot + f * 1024 + lane * 16));
    } else {
        const int blk = (r - 64) >> 1, kb = r & 1;
        char* dst = dbase + OPER_BYTES + (size_t)kt * KT_BYTES + (size_t)blk * HTB + kb * 1024 + lane * 16;
        const int i16 = lane & 15, g = lane >> 4;
#pragma unroll
        for (int nh = 0; nh < 2; ++nh)
#pragma unroll
            for (int q = 0; q < 4; ++q) {
                const int cl = (q >> 1) * 32 + 8 * (i16 >> 2) + 4 * (q & 1) + (i16 & 3);
                const LAS unsigned short* src = (const LAS unsigned short*)(slot + nh * 4096 + (g * 8) * 128 + (((cl >> 3) ^ ((g & 1) << 2)) * 16) + (cl & 7) * 2);
                u32x4 o;
                o.x = (unsigned)src[0 * 64] | ((unsigned)src[1 * 64] << 16);
                o.y = (unsigned)src[2 * 64] | ((unsigned)src[3 * 64] << 16);
                o.z = (unsigned)src[4 * 64] | ((unsigned)src[5 * 64] << 16);
                o.w = (unsigned)src[6 * 64] | ((unsigned)src[7 * 64] << 16);
                st_sys(dst + (nh * 4 + q) * 2048, o);
            }
    }
}
constexpr int U1 = 128 * SPLIT, U2 = U1 + 96 * (8 - SPLIT), U3 = 1024;
__device__ __forceinline__ void unit_decode(int u, int& s, int& r) {
    if (u < U1) { s = u >> 7; r = u & 127; }
    else if (u < U2) { const int v = u - U1; s = SPLIT + v / 96; r = v % 96; }
    else { const int v = u - U2; s = SPLIT + (v >> 5); r = 96 + (v & 31); }
}

__global__ __launch_bounds__(512, 2) void fused_gemm(Params P) {
    extern __shared__ __attribute__((aligned(16))) unsigned char shm[];
    LAS unsigned char* lds = (LAS unsigned char*)shm;
    const int tid = threadIdx.x, wid = __builtin_amdgcn_readfirstlane(tid >> 6), lane = tid & 63;
    const int wr = wid >> 2, wc = wid & 3, fr = lane & 15, fq = lane >> 4;
    const int b = blockIdx.x, xcd = b & 7, idx = b >> 3;
    const int rank = P.rank;
    unsigned* ctl = (unsigned*)P.ws[rank];
    unsigned* tmo = ctl + W_TMO;
    STAMP(0);
    if (tid == 0) {
        const unsigned t = __hip_atomic_fetch_add(ctl + W_TICKET, 1u, __ATOMIC_RELAXED, __HIP_MEMORY_SCOPE_AGENT);
        *(LAS unsigned*)(lds + STAGE_BYTES) = t / gridDim.x + 1u;
    }
    __syncthreads();
    const unsigned call = (unsigned)__builtin_amdgcn_readfirstlane(*(LAS unsigned*)(lds + STAGE_BYTES));
    __syncthreads();
    const size_t bufoff = CTL_BYTES + (size_t)(call & 1u) * BUF_BYTES;
    const int dest = (rank + xcd) & 7;
    const int pm = dest * 2 + (idx & 1), pn = idx >> 1;
    const int half = idx >> 4;
    const int grp = half ? ((idx - 16) >> 2) : 4;
    const int sh0 = grp < 3 ? grp : -1, sh1 = -1;
    const int member = idx & 3;
    auto hook_of = [](int sh) { return sh < 0 ? 8 : (sh == 0 ? 0 : (sh == 1 ? SPLIT - 2 : SPLIT)); };

    auto stage_wait = [&](int h) {
        if (wid == 0) {
            STAMP(16 + h);
            wait_ge(ctl + W_CNT + 32 * h, call * 512u, tmo, call);
            wait_ge(ctl + W_CNT + 32 * ((1 + half) * 8 + h), call * 256u, tmo, call);
            STAMP(1 + h);
            __builtin_amdgcn_fence(__ATOMIC_ACQUIRE, "");
            asm volatile("s_waitcnt vmcnt(0)" ::: "memory");
        }
        asm volatile("" ::: "memory"); __builtin_amdgcn_s_barrier(); asm volatile("" ::: "memory");
    };
    auto do_duty = [&](int shift) {
        int lz = threadIdx.x; asm volatile("" : "+v"(lz)); lz &= 63;
        LAS unsigned* tick = (LAS unsigned*)(lds + STAGE_BYTES + 16);
        if (tid == 0) *tick = 0u;
        STAMP(15);
        if (shift > 0 && wid == 0) wait_ge(ctl + W_DUTY + 32 * ((shift - 1) * 8 + xcd), call * (unsigned)NWG, tmo, call);
        asm volatile("s_waitcnt vmcnt(0) lgkmcnt(0)" ::: "memory"); __builtin_amdgcn_s_barrier(); asm volatile("" ::: "memory");
        STAMP(9);
        const int ubeg = shift == 0 ? 0 : (shift == 1 ? U1 : U2), uend = shift == 0 ? U1 : (shift == 1 ? U2 : U3);
        const int T = (uend - ubeg) / NWG;
        unsigned* dctl = (unsigned*)P.ws[dest];
        char* dbase = P.ws[dest] + bufoff;
        LAS unsigned char* slot0 = lds + wid * 16384;
        auto ticket = [&]() {
            unsigned t = 0;
            if (lz == 0) asm volatile("ds_add_rtn_u32 %0, %1, %2\n\ts_waitcnt lgkmcnt(0)" : "=&v"(t) : "v"((unsigned)(STAGE_BYTES + 16)), "v"(1u) : "memory");
            return (int)__builtin_amdgcn_readfirstlane(t); };
        auto unit_of = [&](int j) { return ubeg + (j >> 3) * (NWG * 8) + member * 8 + (j & 7); };
        int j = ticket(), s_, r, k = 0;
        if (j < T) {
            unsigned* prevc = nullptr;
            unit_decode(unit_of(j), s_, r);
            unit_load(P, s_, dest, r, lz, slot0);
            for (;;) {
                const int jn = ticket(); const bool more = jn < T; int sn, rn;
                unit_decode(unit_of(more ? jn : j), sn, rn);
                unit_load(P, sn, dest, rn, lz, slot0 + ((k + 1) & 1) * 8192);
                if (k == 0) asm volatile("s_waitcnt vmcnt(8)" ::: "memory"); else asm volatile("s_waitcnt vmcnt(16)" ::: "memory");
                unit_store(dbase, s_ * 8 + rank, r, slot0 + (k & 1) * 8192, lz);
                asm volatile("s_waitcnt vmcnt(16) lgkmcnt(0)" ::: "memory");
                if (lz == 0) {
                    if (prevc != nullptr) __hip_atomic_fetch_add(prevc, 1u, __ATOMIC_RELAXED, __HIP_MEMORY_SCOPE_SYSTEM);
                    if (j == T - HANDOFF)
                        __hip_atomic_fetch_add(ctl + W_DUTY + 32 * (shift * 8 + xcd), 1u, __ATOMIC_RELAXED, __HIP_MEMORY_SCOPE_AGENT);
                }
                prevc = dctl + W_CNT + 32 * ((r < 64 ? 0 : (r < 96 ? 1 : 2)) * 8 + s_);
                if (!more) break;
                j = jn; s_ = sn; r = rn; ++k;
            }
            asm volatile("s_waitcnt vmcnt(0)" ::: "memory");
            if (lz == 0) __hip_atomic_fetch_add(prevc, 1u, __ATOMIC_RELAXED, __HIP_MEMORY_SCOPE_SYSTEM);
        }
        STAMP(10);
        asm volatile("" ::: "memory"); __builtin_amdgcn_s_barrier(); asm volatile("" ::: "memory");
    };

    const unsigned voff0 = (unsigned)tid * 16u, voff1 = voff0 + 8192u;
    const unsigned ldsw = (unsigned)wid * 1024u;
    const int aoff = wr * 8192 + lane * 16, boff = wc * 4096 + lane * 16;
#define SA(b_, h_) (((b_) * 2 + (h_)) * HTB)
#define SB(b_, h_) ((4 + (b_) * 2 + (h_)) * HTB)
#define STAGE(bufo, gbase) do { \
        __builtin_amdgcn_global_load_lds((const unsigned*)((const char*)(gbase) + voff0), (LAS unsigned*)(lds + (bufo) + ldsw), 16, 0, 0); \
        __builtin_amdgcn_global_load_lds((const unsigned*)((const char*)(gbase) + voff1), (LAS unsigned*)(lds + (bufo) + ldsw + 8192), 16, 0, 0); } while (0)
#define LDA(dst, b_, h_) do { _Pragma("unroll") for (int m = 0; m < 4; ++m) _Pragma("unroll") for (int k = 0; k < 2; ++k) dst[m][k] = *(const LAS bf16x8*)(lds + SA(b_, h_) + aoff + m * 2048 + k * 1024); } while (0)
#define LDB(dst, b_, h_) do { _Pragma("unroll") for (int n = 0; n < 2; ++n) _Pragma("unroll") for (int k = 0; k < 2; ++k) dst[n][k] = *(const LAS bf16x8*)(lds + SB(b_, h_) + boff + n * 2048 + k * 1024); } while (0)
#define MMA(ai, bj, At_, Bt_) do { __builtin_amdgcn_s_setprio(1); _Pragma("unroll") for (int m = 0; m < 4; ++m) _Pragma("unroll") for (int n = 0; n < 2; ++n) _Pragma("unroll") for (int k = 0; k < 2; ++k) \
        acc[ai][bj][m][n] = __builtin_amdgcn_mfma_f32_16x16x32_bf16(Bt_[n][k], At_[m][k], acc[ai][bj][m][n], 0, 0, 0); __builtin_amdgcn_s_setprio(0); } while (0)
#define WAIT_V(n) asm volatile("s_waitcnt vmcnt(" #n ")" ::: "memory")
#define WAIT_L(n) asm volatile("s_waitcnt lgkmcnt(" #n ")" ::: "memory")
#define BAR __builtin_amdgcn_s_barrier()
#define SCHED __builtin_amdgcn_sched_barrier(0)

    f32x4 acc[2][2][4][2];
#pragma unroll
    for (int a = 0; a < 2; ++a)
#pragma unroll
        for (int c = 0; c < 2; ++c)
#pragma unroll
            for (int m = 0; m < 4; ++m)
#pragma unroll
                for (int n = 0; n < 2; ++n) acc[a][c][m][n] = (f32x4){0.f, 0.f, 0.f, 0.f};
    bf16x8 At[4][2], B0[2][2], B1[2][2];
    const char* cA = P.ws[rank] + bufoff + (size_t)pm * 2 * HTB;
    const char* cB = P.ws[rank] + bufoff + OPER_BYTES + (size_t)pn * 2 * HTB;
    constexpr size_t kstep = KT_BYTES, hstep = HTB;

    int kt0 = 0;
    for (int seg = 0; seg < 3; ++seg) {
        const int sh = seg == 0 ? sh0 : (seg == 1 ? sh1 : -1);
        const int kt1 = hook_of(sh) * 8;
        if (kt1 > kt0) {
            stage_wait(kt0 >> 3);
            {
                const char* a0 = cA + (size_t)kt0 * kstep; const char* b0 = cB + (size_t)kt0 * kstep;
                STAGE(SB(0, 0), b0); STAGE(SA(0, 0), a0); STAGE(SB(0, 1), b0 + hstep); STAGE(SA(0, 1), a0 + hstep);
                if (wr == 1) BAR;
                WAIT_V(4); BAR;
                STAGE(SB(1, 0), b0 + kstep); STAGE(SA(1, 0), a0 + kstep); STAGE(SB(1, 1), b0 + hstep + kstep);
                WAIT_V(6); BAR;
            }
            for (int t = kt0; t < kt1; t += 2) {
                const bool last = (t == kt1 - 2);
                const char* a1 = cA + (size_t)(t + 1) * kstep;
                const char* a2 = last ? cA : cA + (size_t)(t + 2) * kstep; const char* b2 = last ? cB : cB + (size_t)(t + 2) * kstep;
                const char* a3 = a2 + kstep; const char* b3 = b2 + kstep;
                if (((t + 2) & 7) == 0 && !last) stage_wait((t + 2) >> 3);
                LDB(B0, 0, 0); SCHED; LDA(At, 0, 0); STAGE(SA(1, 1), a1 + hstep);
                WAIT_L(8); BAR; WAIT_L(0); MMA(0, 0, At, B0); BAR; SCHED;
                LDB(B1, 0, 1); STAGE(SB(0, 0), b2);
                BAR; WAIT_L(0); MMA(0, 1, At, B1); BAR;
                LDA(At, 0, 1); STAGE(SA(0, 0), a2);
                BAR; WAIT_L(0); MMA(1, 0, At, B0); BAR; SCHED;
                STAGE(SB(0, 1), b2 + hstep);
                WAIT_V(6); BAR; MMA(1, 1, At, B1); BAR;
                LDB(B0, 1, 0); SCHED; LDA(At, 1, 0); STAGE(SA(0, 1), a2 + hstep);
                WAIT_L(8); BAR; WAIT_L(0); MMA(0, 0, At, B0); BAR; SCHED;
                LDB(B1, 1, 1); STAGE(SB(1, 0), b3);
                BAR; WAIT_L(0); MMA(0, 1, At, B1); BAR;
                LDA(At, 1, 1); STAGE(SA(1, 0), a3);
                BAR; WAIT_L(0); MMA(1, 0, At, B0); BAR; SCHED;
                STAGE(SB(1, 1), b3 + hstep);
                WAIT_V(6); BAR; MMA(1, 1, At, B1); BAR;
            }
            WAIT_V(0);
            if (wr == 0) BAR;
            BAR;
        }
        if (sh >= 0) do_duty(sh);
        kt0 = kt1;
    }
    STAMP(11);
    {
        const bool bad = (unsigned)__builtin_amdgcn_readfirstlane(__hip_atomic_load(tmo, __ATOMIC_RELAXED, __HIP_MEMORY_SCOPE_AGENT)) == call;
        char* obase = P.out[dest] + ((size_t)((pm & 1) * 256 + wr * 64 + fr) * NTOT + (size_t)rank * NSL + pn * 256 + wc * 32 + 8 * fq) * 2;
#pragma unroll
        for (int ai = 0; ai < 2; ++ai)
#pragma unroll
            for (int m = 0; m < 4; ++m) {
                char* rowp = obase + (size_t)(ai * 128 + m * 16) * NTOT * 2;
#pragma unroll
                for (int bj = 0; bj < 2; ++bj) {
                    const f32x4 v0 = acc[ai][bj][m][0], v1 = acc[ai][bj][m][1];
                    u32x4 o;
                    o.x = cvt_pk_bf16(gelu1(v0[0]), gelu1(v0[1])); o.y = cvt_pk_bf16(gelu1(v0[2]), gelu1(v0[3]));
                    o.z = cvt_pk_bf16(gelu1(v1[0]), gelu1(v1[1])); o.w = cvt_pk_bf16(gelu1(v1[2]), gelu1(v1[3]));
                    if (bad) o = (u32x4){0x7fc07fc0u, 0x7fc07fc0u, 0x7fc07fc0u, 0x7fc07fc0u};
                    st_sys(rowp + bj * 256, o);
                }
            }
    }
    STAMP(12);
    asm volatile("s_waitcnt vmcnt(0)" ::: "memory");
    __syncthreads();
    if (tid == 0) {
        __hip_atomic_fetch_add((unsigned*)P.ws[dest] + W_OUT, 1u, __ATOMIC_RELAXED, __HIP_MEMORY_SCOPE_SYSTEM);
    }
    STAMP(13);
    if (b == 0 && wid == 0) {
        wait_ge(ctl + W_OUT, call * 256u, tmo, call);
        STAMP(14);
        __builtin_amdgcn_fence(__ATOMIC_ACQUIRE, "");
        if ((unsigned)__builtin_amdgcn_readfirstlane(__hip_atomic_load(tmo, __ATOMIC_RELAXED, __HIP_MEMORY_SCOPE_AGENT)) == call && lane == 0)
            *(unsigned short*)P.out[rank] = (unsigned short)0x7fc0u;
    }
}
}

extern "C" void kernel_launch(int rank, int nranks, ncclComm_t comm,
                              hipStream_t stream,
                              void* const* d_in, const int* in_sizes, int n_in,
                              void* d_out, int out_size,
                              const int* in_dtypes, int out_dtype,
                              void* const* d_out_all,
                              void* const* d_ws_all, size_t ws_bytes) {
    if (nranks != NR || ws_bytes < WS_NEED || n_in < 2) { fprintf(stderr, "unexpected configuration\n"); return; }
    static bool attr = false;
    if (!attr) { HIP_CHECK(hipFuncSetAttribute((const void*)fused_gemm, hipFuncAttributeMaxDynamicSharedMemorySize, LDS_BYTES)); attr = true; }
    Params P;
    P.x = (const bf16_t*)d_in[0]; P.w = (const bf16_t*)d_in[1]; P.rank = rank;
    for (int p = 0; p < NR; ++p) { P.ws[p] = (char*)d_ws_all[p]; P.out[p] = (char*)d_out_all[p]; }
    hipLaunchKernelGGL(fused_gemm, dim3(256), dim3(512), LDS_BYTES, stream, P);
    HIP_CHECK(hipGetLastError());
}
```

```cpp
#include <hip/hip_runtime.h>
#include <rccl/rccl.h>
#include <cstdio>
#include <cstdint>

#define HIP_CHECK(expr) do { hipError_t _e = (expr); if (_e != hipSuccess) { \
    fprintf(stderr, "HIP error %s at %s:%d\n", hipGetErrorString(_e), __FILE__, __LINE__); \
    return; } } while (0)

namespace {
#define LAS __attribute__((address_space(3)))
typedef unsigned short bf16_t;
typedef short bf16x8 __attribute__((ext_vector_type(8)));
typedef float f32x4 __attribute__((ext_vector_type(4)));
typedef unsigned u32x4 __attribute__((ext_vector_type(4)));

#define SPLIT 4
#define NWG 8
#define PW 2
#define HANDOFF 8
#define RELFENCE 0
constexpr int NR = 8;
constexpr int KS = 512;
constexpr int NTOT = 32768;
constexpr int NSL = 4096;
constexpr int NKT = 64;
constexpr int HTB = 16384;
constexpr int STAGE_BYTES = 8 * HTB;
constexpr int SCR_BYTES = 256;
constexpr int LDS_BYTES = STAGE_BYTES + SCR_BYTES;
constexpr size_t CTL_BYTES = 1u << 20;
constexpr size_t OPER_BYTES = 32u << 20;
constexpr size_t BUF_BYTES = 2 * OPER_BYTES;
constexpr size_t KT_BYTES = 32 * (size_t)HTB;
constexpr size_t WS_NEED = CTL_BYTES + 2 * BUF_BYTES;
constexpr int W_TICKET = 0, W_TMO = 32, W_OUT = 64, W_CNT = 128  , W_DUTY = 1024  ;
constexpr unsigned long long DEADLINE = 100000000ull;

#define STAMP(i) do { } while (0)

struct Params {
    const bf16_t* x; const bf16_t* w;
    char* ws[NR]; char* out[NR];
    int rank;
};

__device__ __forceinline__ void st_sys(void* p, u32x4 v) {
    asm volatile("global_store_dwordx4 %0, %1, off sc0 sc1\n\ts_nop 1" :: "v"(p), "v"(v) : "memory");
}
__device__ __forceinline__ unsigned cvt_pk_bf16(float lo, float hi) { unsigned r; asm volatile("v_cvt_pk_bf16_f32 %0, %1, %2" : "=v"(r) : "v"(lo), "v"(hi)); return r; }
__device__ __forceinline__ float gelu1(float y) {
    const float k1 = -2.302208198144325f;
    const float k3 = k1 * 0.044715f;
    const float t = y * (k1 + k3 * y * y);
    const float e = __builtin_amdgcn_exp2f(t);
    return y * __builtin_amdgcn_rcpf(1.0f + e);
}

__device__ __forceinline__ void wait_ge(const unsigned* p, unsigned need, unsigned* tmo, unsigned call) {
    const unsigned long long t0 = __builtin_amdgcn_s_memrealtime();
    while ((unsigned)__builtin_amdgcn_readfirstlane(__hip_atomic_load(p, __ATOMIC_RELAXED, __HIP_MEMORY_SCOPE_SYSTEM)) < need) {
        if ((unsigned)__builtin_amdgcn_readfirstlane(__hip_atomic_load(tmo, __ATOMIC_RELAXED, __HIP_MEMORY_SCOPE_AGENT)) == call
            || __builtin_amdgcn_s_memrealtime() - t0 > DEADLINE) {
            __hip_atomic_store(tmo, call, __ATOMIC_RELAXED, __HIP_MEMORY_SCOPE_AGENT);
            break;
        }
        __builtin_amdgcn_s_sleep(2);
    }
}

__device__ __forceinline__ void unit_load(const Params& P, int s, int d, int r, int lane, LAS unsigned char* slot) {
    const bool isA = r < 64;
    const int blk = (r - 64) >> 1, kb = r & 1;
    const char* base = isA ? (const char*)(P.x + (size_t)(r * 64) * KS + s * 64) : (const char*)(P.w + (size_t)(s * 64 + kb * 32) * NTOT + d * NSL + blk * 128);
    const unsigned l0 = isA ? (unsigned)((lane & 15) * KS + (lane >> 4) * 8) * 2u : (unsigned)((lane >> 3) * NTOT + (lane & 7) * 8) * 2u;
    const unsigned l1 = isA ? l0 : (unsigned)((lane >> 3) * NTOT + ((lane & 7) ^ 4) * 8) * 2u;
    const unsigned s1 = isA ? 16u * KS * 2u : 8u * NTOT * 2u, s2 = isA ? 64u : 128u;
    const unsigned d1 = isA ? 2048u : 1024u, d2 = isA ? 1024u : 4096u;
#pragma unroll
    for (int f = 0; f < 8; ++f)
        __builtin_amdgcn_global_load_lds((const unsigned*)(base + ((((f >> 1) & 1) ? l1 : l0) + (f >> 1) * s1 + (f & 1) * s2)),
                                         (LAS unsigned*)(slot + (f >> 1) * d1 + (f & 1) * d2), 16, 0, 0);
}
__device__ __forceinline__ void unit_store(char* dbase, int kt, int r, const LAS unsigned char* slot, int lane) {
    if (r < 64) {
        char* dst = dbase + (size_t)kt * KT_BYTES + (size_t)r * (HTB / 2) + lane * 16;
#pragma unroll
        for (int f = 0; f < 8; ++f) st_sys(dst + f * 1024, *(const LAS u32x4*)(slot + f * 1024 + lane * 16));
    } else {
        const int blk = (r - 64) >> 1, kb = r & 1;
        char* dst = dbase + OPER_BYTES + (size_t)kt * KT_BYTES + (size_t)blk * HTB + kb * 1024 + lane * 16;
        const int i16 = lane & 15, g = lane >> 4;
#pragma unroll
        for (int nh = 0; nh < 2; ++nh)
#pragma unroll
            for (int q = 0; q < 4; ++q) {
                const int cl = (q >> 1) * 32 + 8 * (i16 >> 2) + 4 * (q & 1) + (i16 & 3);
                const LAS unsigned short* src = (const LAS unsigned short*)(slot + nh * 4096 + (g * 8) * 128 + (((cl >> 3) ^ ((g & 1) << 2)) * 16) + (cl & 7) * 2);
                u32x4 o;
                o.x = (unsigned)src[0 * 64] | ((unsigned)src[1 * 64] << 16);
                o.y = (unsigned)src[2 * 64] | ((unsigned)src[3 * 64] << 16);
                o.z = (unsigned)src[4 * 64] | ((unsigned)src[5 * 64] << 16);
                o.w = (unsigned)src[6 * 64] | ((unsigned)src[7 * 64] << 16);
                st_sys(dst + (nh * 4 + q) * 2048, o);
            }
    }
}
constexpr int U1 = 128 * SPLIT, U2 = U1 + 96 * (8 - SPLIT), U3 = 1024;
static_assert(U1 % (NWG * 8) == 0 && (U2 - U1) % (NWG * 8) == 0 && (U3 - U2) % (NWG * 8) == 0, "shift sizes must divide over the duty workgroups");
__device__ __forceinline__ void unit_decode(int u, int& s, int& r) {
    if (u < U1) { s = u >> 7; r = u & 127; }
    else if (u < U2) { const int v = u - U1; s = SPLIT + v / 96; r = v % 96; }
    else { const int v = u - U2; s = SPLIT + (v >> 5); r = 96 + (v & 31); }
}

__global__ __launch_bounds__(512, 2) void fused_gemm(Params P) {
    extern __shared__ __attribute__((aligned(16))) unsigned char shm[];
    LAS unsigned char* lds = (LAS unsigned char*)shm;
    const int tid = threadIdx.x, wid = __builtin_amdgcn_readfirstlane(tid >> 6), lane = tid & 63;
    const int wr = wid >> 2, wc = wid & 3, fr = lane & 15, fq = lane >> 4;
    const int b = blockIdx.x, xcd = b & 7, idx = b >> 3;
    const int rank = P.rank;
    unsigned* ctl = (unsigned*)P.ws[rank];
    unsigned* tmo = ctl + W_TMO;
    STAMP(0);
    if (tid == 0) {
        const unsigned t = __hip_atomic_fetch_add(ctl + W_TICKET, 1u, __ATOMIC_RELAXED, __HIP_MEMORY_SCOPE_AGENT);
        *(LAS unsigned*)(lds + STAGE_BYTES) = t / gridDim.x + 1u;
    }
    __syncthreads();
    const unsigned call = (unsigned)__builtin_amdgcn_readfirstlane(*(LAS unsigned*)(lds + STAGE_BYTES));
    __syncthreads();
    const size_t bufoff = CTL_BYTES + (size_t)(call & 1u) * BUF_BYTES;
    const int dest = (rank + xcd) & 7;
    const int pm = dest * 2 + (idx & 1), pn = idx >> 1;
    const int half = idx >> 4;
    const int grp = half ? ((idx - 16) >> 2) : 4;
    const int sh0 = grp < 2 ? 0 : (grp < 4 ? 1 : -1), sh1 = grp < 2 ? 2 : -1;
    const int member = idx & 7;
    auto hook_of = [](int sh) { return sh < 0 ? 8 : (sh == 0 ? 0 : (sh == 1 ? SPLIT - 2 : SPLIT)); };

    auto stage_wait = [&](int h) {
        if (wid == 0) {
            STAMP(16 + h);
            wait_ge(ctl + W_CNT + 32 * h, call * 512u, tmo, call);
            wait_ge(ctl + W_CNT + 32 * ((1 + half) * 8 + h), call * 256u, tmo, call);
            STAMP(1 + h);
            __builtin_amdgcn_fence(__ATOMIC_ACQUIRE, "");
            asm volatile("s_waitcnt vmcnt(0)" ::: "memory");
        }
        asm volatile("" ::: "memory"); __builtin_amdgcn_s_barrier(); asm volatile("" ::: "memory");
    };
    auto do_duty = [&](int shift) {
        int lz = threadIdx.x; asm volatile("" : "+v"(lz)); lz &= 63;
        LAS unsigned* tick = (LAS unsigned*)(lds + STAGE_BYTES + 16);
        if (tid == 0) *tick = 0u;
        STAMP(15);
        if (shift > 0 && wid == 0) wait_ge(ctl + W_DUTY + 32 * ((shift - 1) * 8 + xcd), call * (unsigned)NWG, tmo, call);
        asm volatile("s_waitcnt vmcnt(0) lgkmcnt(0)" ::: "memory"); __builtin_amdgcn_s_barrier(); asm volatile("" ::: "memory");
        STAMP(9);
        const int ubeg = shift == 0 ? 0 : (shift == 1 ? U1 : U2), uend = shift == 0 ? U1 : (shift == 1 ? U2 : U3);
        const int T = (uend - ubeg) / NWG;
        unsigned* dctl = (unsigned*)P.ws[dest];
        char* dbase = P.ws[dest] + bufoff;
        LAS unsigned char* slot0 = lds + wid * 16384;
        auto ticket = [&]() {
            unsigned t = 0;
            if (lz == 0) asm volatile("ds_add_rtn_u32 %0, %1, %2\n\ts_waitcnt lgkmcnt(0)" : "=&v"(t) : "v"((unsigned)(STAGE_BYTES + 16)), "v"(1u) : "memory");
            return (int)__builtin_amdgcn_readfirstlane(t); };
        auto unit_of = [&](int j) { return ubeg + (j >> 3) * (NWG * 8) + member * 8 + (j & 7); };
        int j = wid < PW ? ticket() : T, s_, r, k = 0;
        if (j < T) {
            unsigned* prevc = nullptr;
            unit_decode(unit_of(j), s_, r);
            unit_load(P, s_, dest, r, lz, slot0);
            for (;;) {
                const int jn = ticket(); const bool more = jn < T; int sn, rn;
                unit_decode(unit_of(more ? jn : j), sn, rn);
                unit_load(P, sn, dest, rn, lz, slot0 + ((k + 1) & 1) * 8192);
                if (k == 0) asm volatile("s_waitcnt vmcnt(8)" ::: "memory"); else asm volatile("s_waitcnt vmcnt(16)" ::: "memory");
                unit_store(dbase, s_ * 8 + rank, r, slot0 + (k & 1) * 8192, lz);
                asm volatile("s_waitcnt vmcnt(16) lgkmcnt(0)" ::: "memory");
                if (lz == 0) {
                    if (prevc != nullptr) __hip_atomic_fetch_add(prevc, 1u, __ATOMIC_RELAXED, __HIP_MEMORY_SCOPE_SYSTEM);
                    if (j == T - HANDOFF)
                        __hip_atomic_fetch_add(ctl + W_DUTY + 32 * (shift * 8 + xcd), 1u, __ATOMIC_RELAXED, __HIP_MEMORY_SCOPE_AGENT);
                }
                prevc = dctl + W_CNT + 32 * ((r < 64 ? 0 : (r < 96 ? 1 : 2)) * 8 + s_);
                if (!more) break;
                j = jn; s_ = sn; r = rn; ++k;
            }
            asm volatile("s_waitcnt vmcnt(0)" ::: "memory");
            if (lz == 0) __hip_atomic_fetch_add(prevc, 1u, __ATOMIC_RELAXED, __HIP_MEMORY_SCOPE_SYSTEM);
        }
        STAMP(10);
        asm volatile("" ::: "memory"); __builtin_amdgcn_s_barrier(); asm volatile("" ::: "memory");
    };

    const unsigned voff0 = (unsigned)tid * 16u, voff1 = voff0 + 8192u;
    const unsigned ldsw = (unsigned)wid * 1024u;
    const int aoff = wr * 8192 + lane * 16, boff = wc * 4096 + lane * 16;
#define SA(b_, h_) (((b_) * 2 + (h_)) * HTB)
#define SB(b_, h_) ((4 + (b_) * 2 + (h_)) * HTB)
#define STAGE(bufo, gbase) do { \
        __builtin_amdgcn_global_load_lds((const unsigned*)((const char*)(gbase) + voff0), (LAS unsigned*)(lds + (bufo) + ldsw), 16, 0, 0); \
        __builtin_amdgcn_global_load_lds((const unsigned*)((const char*)(gbase) + voff1), (LAS unsigned*)(lds + (bufo) + ldsw + 8192), 16, 0, 0); } while (0)
#define LDA(dst, b_, h_) do { _Pragma("unroll") for (int m = 0; m < 4; ++m) _Pragma("unroll") for (int k = 0; k < 2; ++k) dst[m][k] = *(const LAS bf16x8*)(lds + SA(b_, h_) + aoff + m * 2048 + k * 1024); } while (0)
#define LDB(dst, b_, h_) do { _Pragma("unroll") for (int n = 0; n < 2; ++n) _Pragma("unroll") for (int k = 0; k < 2; ++k) dst[n][k] = *(const LAS bf16x8*)(lds + SB(b_, h_) + boff + n * 2048 + k * 1024); } while (0)
#define MMA(ai, bj, At_, Bt_) do { __builtin_amdgcn_s_setprio(1); _Pragma("unroll") for (int m = 0; m < 4; ++m) _Pragma("unroll") for (int n = 0; n < 2; ++n) _Pragma("unroll") for (int k = 0; k < 2; ++k) \
        acc[ai][bj][m][n] = __builtin_amdgcn_mfma_f32_16x16x32_bf16(Bt_[n][k], At_[m][k], acc[ai][bj][m][n], 0, 0, 0); __builtin_amdgcn_s_setprio(0); } while (0)
#define WAIT_V(n) asm volatile("s_waitcnt vmcnt(" #n ")" ::: "memory")
#define WAIT_L(n) asm volatile("s_waitcnt lgkmcnt(" #n ")" ::: "memory")
#define BAR __builtin_amdgcn_s_barrier()
#define SCHED __builtin_amdgcn_sched_barrier(0)

    f32x4 acc[2][2][4][2];
#pragma unroll
    for (int a = 0; a < 2; ++a)
#pragma unroll
        for (int c = 0; c < 2; ++c)
#pragma unroll
            for (int m = 0; m < 4; ++m)
#pragma unroll
                for (int n = 0; n < 2; ++n) acc[a][c][m][n] = (f32x4){0.f, 0.f, 0.f, 0.f};
    bf16x8 At[4][2], B0[2][2], B1[2][2];
    const char* cA = P.ws[rank] + bufoff + (size_t)pm * 2 * HTB;
    const char* cB = P.ws[rank] + bufoff + OPER_BYTES + (size_t)pn * 2 * HTB;
    constexpr size_t kstep = KT_BYTES, hstep = HTB;

    int kt0 = 0;
    for (int seg = 0; seg < 3; ++seg) {
        const int sh = seg == 0 ? sh0 : (seg == 1 ? sh1 : -1);
        const int kt1 = hook_of(sh) * 8;
        if (kt1 > kt0) {
            stage_wait(kt0 >> 3);
            {
                const char* a0 = cA + (size_t)kt0 * kstep; const char* b0 = cB + (size_t)kt0 * kstep;
                STAGE(SB(0, 0), b0); STAGE(SA(0, 0), a0); STAGE(SB(0, 1), b0 + hstep); STAGE(SA(0, 1), a0 + hstep);
                if (wr == 1) BAR;
                WAIT_V(4); BAR;
                STAGE(SB(1, 0), b0 + kstep); STAGE(SA(1, 0), a0 + kstep); STAGE(SB(1, 1), b0 + hstep + kstep);
                WAIT_V(6); BAR;
            }
            for (int t = kt0; t < kt1; t += 2) {
                const bool last = (t == kt1 - 2);
                const char* a1 = cA + (size_t)(t + 1) * kstep;
                const char* a2 = last ? cA : cA + (size_t)(t + 2) * kstep; const char* b2 = last ? cB : cB + (size_t)(t + 2) * kstep;
                const char* a3 = a2 + kstep; const char* b3 = b2 + kstep;
                if (((t + 2) & 7) == 0 && !last) stage_wait((t + 2) >> 3);
                LDB(B0, 0, 0); SCHED; LDA(At, 0, 0); STAGE(SA(1, 1), a1 + hstep);
                WAIT_L(8); BAR; WAIT_L(0); MMA(0, 0, At, B0); BAR; SCHED;
                LDB(B1, 0, 1); STAGE(SB(0, 0), b2);
                BAR; WAIT_L(0); MMA(0, 1, At, B1); BAR;
                LDA(At, 0, 1); STAGE(SA(0, 0), a2);
                BAR; WAIT_L(0); MMA(1, 0, At, B0); BAR; SCHED;
                STAGE(SB(0, 1), b2 + hstep);
                WAIT_V(6); BAR; MMA(1, 1, At, B1); BAR;
                LDB(B0, 1, 0); SCHED; LDA(At, 1, 0); STAGE(SA(0, 1), a2 + hstep);
                WAIT_L(8); BAR; WAIT_L(0); MMA(0, 0, At, B0); BAR; SCHED;
                LDB(B1, 1, 1); STAGE(SB(1, 0), b3);
                BAR; WAIT_L(0); MMA(0, 1, At, B1); BAR;
                LDA(At, 1, 1); STAGE(SA(1, 0), a3);
                BAR; WAIT_L(0); MMA(1, 0, At, B0); BAR; SCHED;
                STAGE(SB(1, 1), b3 + hstep);
                WAIT_V(6); BAR; MMA(1, 1, At, B1); BAR;
            }
            WAIT_V(0);
            if (wr == 0) BAR;
            BAR;
        }
        if (sh >= 0) do_duty(sh);
        kt0 = kt1;
    }
    STAMP(11);
    {
        const bool bad = (unsigned)__builtin_amdgcn_readfirstlane(__hip_atomic_load(tmo, __ATOMIC_RELAXED, __HIP_MEMORY_SCOPE_AGENT)) == call;
        char* obase = P.out[dest] + ((size_t)((pm & 1) * 256 + wr * 64 + fr) * NTOT + (size_t)rank * NSL + pn * 256 + wc * 32 + 8 * fq) * 2;
#pragma unroll
        for (int ai = 0; ai < 2; ++ai)
#pragma unroll
            for (int m = 0; m < 4; ++m) {
                char* rowp = obase + (size_t)(ai * 128 + m * 16) * NTOT * 2;
#pragma unroll
                for (int bj = 0; bj < 2; ++bj) {
                    const f32x4 v0 = acc[ai][bj][m][0], v1 = acc[ai][bj][m][1];
                    u32x4 o;
                    o.x = cvt_pk_bf16(gelu1(v0[0]), gelu1(v0[1])); o.y = cvt_pk_bf16(gelu1(v0[2]), gelu1(v0[3]));
                    o.z = cvt_pk_bf16(gelu1(v1[0]), gelu1(v1[1])); o.w = cvt_pk_bf16(gelu1(v1[2]), gelu1(v1[3]));
                    if (bad) o = (u32x4){0x7fc07fc0u, 0x7fc07fc0u, 0x7fc07fc0u, 0x7fc07fc0u};
                    st_sys(rowp + bj * 256, o);
                }
            }
    }
    STAMP(12);
    asm volatile("s_waitcnt vmcnt(0)" ::: "memory");
    __syncthreads();
    if (tid == 0) {
        __hip_atomic_fetch_add((unsigned*)P.ws[dest] + W_OUT, 1u, __ATOMIC_RELAXED, __HIP_MEMORY_SCOPE_SYSTEM);
    }
    STAMP(13);
    if (b == 0 && wid == 0) {
        wait_ge(ctl + W_OUT, call * 256u, tmo, call);
        STAMP(14);
        __builtin_amdgcn_fence(__ATOMIC_ACQUIRE, "");
        if ((unsigned)__builtin_amdgcn_readfirstlane(__hip_atomic_load(tmo, __ATOMIC_RELAXED, __HIP_MEMORY_SCOPE_AGENT)) == call && lane == 0)
            *(unsigned short*)P.out[rank] = (unsigned short)0x7fc0u;
    }
}
}

extern "C" void kernel_launch(int rank, int nranks, ncclComm_t comm,
                              hipStream_t stream,
                              void* const* d_in, const int* in_sizes, int n_in,
                              void* d_out, int out_size,
                              const int* in_dtypes, int out_dtype,
                              void* const* d_out_all,
                              void* const* d_ws_all, size_t ws_bytes) {
    if (nranks != NR || ws_bytes < WS_NEED || n_in < 2) { fprintf(stderr, "unexpected configuration\n"); return; }
    static bool attr = false;
    if (!attr) { HIP_CHECK(hipFuncSetAttribute((const void*)fused_gemm, hipFuncAttributeMaxDynamicSharedMemorySize, LDS_BYTES)); attr = true; }
    Params P;
    P.x = (const bf16_t*)d_in[0]; P.w = (const bf16_t*)d_in[1]; P.rank = rank;
    for (int p = 0; p < NR; ++p) { P.ws[p] = (char*)d_ws_all[p]; P.out[p] = (char*)d_out_all[p]; }
    hipLaunchKernelGGL(fused_gemm, dim3(256), dim3(512), LDS_BYTES, stream, P);
    HIP_CHECK(hipGetLastError());
}
```

```cpp
#include <hip/hip_runtime.h>
#include <rccl/rccl.h>
#include <cstdio>
#include <cstdint>

#define HIP_CHECK(expr) do { hipError_t _e = (expr); if (_e != hipSuccess) { \
    fprintf(stderr, "HIP error %s at %s:%d\n", hipGetErrorString(_e), __FILE__, __LINE__); \
    return; } } while (0)

namespace {
#define LAS __attribute__((address_space(3)))
typedef unsigned short bf16_t;
typedef short bf16x8 __attribute__((ext_vector_type(8)));
typedef float f32x4 __attribute__((ext_vector_type(4)));
typedef unsigned u32x4 __attribute__((ext_vector_type(4)));

constexpr int SPLIT = 4;
constexpr int NWG = 8;
constexpr int PW = 2;
constexpr int HANDOFF = 8;
constexpr int NR = 8;
constexpr int KS = 512;
constexpr int NTOT = 32768;
constexpr int NSL = 4096;
constexpr int HTB = 16384;
constexpr int STAGE_BYTES = 8 * HTB;
constexpr int SCR_BYTES = 256;
constexpr int LDS_BYTES = STAGE_BYTES + SCR_BYTES;
constexpr size_t CTL_BYTES = 1u << 20;
constexpr size_t OPER_BYTES = 32u << 20;
constexpr size_t BUF_BYTES = 2 * OPER_BYTES;
constexpr size_t KT_BYTES = 32 * (size_t)HTB;
constexpr size_t WS_NEED = CTL_BYTES + 2 * BUF_BYTES;
constexpr int W_TICKET = 0, W_TMO = 32, W_OUT = 64, W_CNT = 128  , W_DUTY = 1024  ;
constexpr unsigned long long DEADLINE = 100000000ull;

struct Params {
    const bf16_t* x; const bf16_t* w;
    char* ws[NR]; char* out[NR];
    int rank;
};

__device__ __forceinline__ void st_sys(void* p, u32x4 v) {
    asm volatile("global_store_dwordx4 %0, %1, off sc0 sc1\n\ts_nop 1" :: "v"(p), "v"(v) : "memory");
}
__device__ __forceinline__ unsigned cvt_pk_bf16(float lo, float hi) { unsigned r; asm volatile("v_cvt_pk_bf16_f32 %0, %1, %2" : "=v"(r) : "v"(lo), "v"(hi)); return r; }
__device__ __forceinline__ float gelu1(float y) {
    const float k1 = -2.302208198144325f;
    const float k3 = k1 * 0.044715f;
    const float t = y * (k1 + k3 * y * y);
    const float e = __builtin_amdgcn_exp2f(t);
    return y * __builtin_amdgcn_rcpf(1.0f + e);
}

__device__ __forceinline__ void wait_ge(const unsigned* p, unsigned need, unsigned* tmo, unsigned call) {
    const unsigned long long t0 = __builtin_amdgcn_s_memrealtime();
    while ((unsigned)__builtin_amdgcn_readfirstlane(__hip_atomic_load(p, __ATOMIC_RELAXED, __HIP_MEMORY_SCOPE_SYSTEM)) < need) {
        if ((unsigned)__builtin_amdgcn_readfirstlane(__hip_atomic_load(tmo, __ATOMIC_RELAXED, __HIP_MEMORY_SCOPE_AGENT)) == call
            || __builtin_amdgcn_s_memrealtime() - t0 > DEADLINE) {
            __hip_atomic_store(tmo, call, __ATOMIC_RELAXED, __HIP_MEMORY_SCOPE_AGENT);
            break;
        }
        __builtin_amdgcn_s_sleep(2);
    }
}

__device__ __forceinline__ void unit_load(const Params& P, int s, int d, int r, int lane, LAS unsigned char* slot) {
    const bool isA = r < 64;
    const int blk = (r - 64) >> 1, kb = r & 1;
    const char* base = isA ? (const char*)(P.x + (size_t)(r * 64) * KS + s * 64) : (const char*)(P.w + (size_t)(s * 64 + kb * 32) * NTOT + d * NSL + blk * 128);
    const unsigned l0 = isA ? (unsigned)((lane & 15) * KS + (lane >> 4) * 8) * 2u : (unsigned)((lane >> 3) * NTOT + (lane & 7) * 8) * 2u;
    const unsigned l1 = isA ? l0 : (unsigned)((lane >> 3) * NTOT + ((lane & 7) ^ 4) * 8) * 2u;
    const unsigned s1 = isA ? 16u * KS * 2u : 8u * NTOT * 2u, s2 = isA ? 64u : 128u;
    const unsigned d1 = isA ? 2048u : 1024u, d2 = isA ? 1024u : 4096u;
#pragma unroll
    for (int f = 0; f < 8; ++f)
        __builtin_amdgcn_global_load_lds((const unsigned*)(base + ((((f >> 1) & 1) ? l1 : l0) + (f >> 1) * s1 + (f & 1) * s2)),
                                         (LAS unsigned*)(slot + (f >> 1) * d1 + (f & 1) * d2), 16, 0, 0);
}
__device__ __forceinline__ void unit_store(char* dbase, int kt, int r, const LAS unsigned char* slot, int lane) {
    if (r < 64) {
        char* dst = dbase + (size_t)kt * KT_BYTES + (size_t)r * (HTB / 2) + lane * 16;
#pragma unroll
        for (int f = 0; f < 8; ++f) st_sys(dst + f * 1024, *(const LAS u32x4*)(slot + f * 1024 + lane * 16));
    } else {
        const int blk = (r - 64) >> 1, kb = r & 1;
        char* dst = dbase + OPER_BYTES + (size_t)kt * KT_BYTES + (size_t)blk * HTB + kb * 1024 + lane * 16;
        const int i16 = lane & 15, g = lane >> 4;
#pragma unroll
        for (int nh = 0; nh < 2; ++nh)
#pragma unroll
            for (int q = 0; q < 4; ++q) {
                const int cl = (q >> 1) * 32 + 8 * (i16 >> 2) + 4 * (q & 1) + (i16 & 3);
                const LAS unsigned short* src = (const LAS unsigned short*)(slot + nh * 4096 + (g * 8) * 128 + (((cl >> 3) ^ ((g & 1) << 2)) * 16) + (cl & 7) * 2);
                u32x4 o;
                o.x = (unsigned)src[0 * 64] | ((unsigned)src[1 * 64] << 16);
                o.y = (unsigned)src[2 * 64] | ((unsigned)src[3 * 64] << 16);
                o.z = (unsigned)src[4 * 64] | ((unsigned)src[5 * 64] << 16);
                o.w = (unsigned)src[6 * 64] | ((unsigned)src[7 * 64] << 16);
                st_sys(dst + (nh * 4 + q) * 2048, o);
            }
    }
}
constexpr int U1 = 128 * SPLIT, U2 = U1 + 96 * (8 - SPLIT), U3 = 1024;
static_assert(U1 % (NWG * 8) == 0 && (U2 - U1) % (NWG * 8) == 0 && (U3 - U2) % (NWG * 8) == 0, "shift sizes must divide over the duty workgroups");
static_assert((U3 - U2) / NWG >= HANDOFF && NWG == 8, "duty layout");
__device__ __forceinline__ void unit_decode(int u, int& s, int& r) {
    if (u < U1) { s = u >> 7; r = u & 127; }
    else if (u < U2) { const int v = u - U1; s = SPLIT + v / 96; r = v % 96; }
    else { const int v = u - U2; s = SPLIT + (v >> 5); r = 96 + (v & 31); }
}

__global__ __launch_bounds__(512, 2) void fused_gemm(Params P) {
    extern __shared__ __attribute__((aligned(16))) unsigned char shm[];
    LAS unsigned char* lds = (LAS unsigned char*)shm;
    const int tid = threadIdx.x, wid = __builtin_amdgcn_readfirstlane(tid >> 6), lane = tid & 63;
    const int wr = wid >> 2, wc = wid & 3, fr = lane & 15, fq = lane >> 4;
    const int b = blockIdx.x, xcd = b & 7, idx = b >> 3;
    const int rank = P.rank;
    unsigned* ctl = (unsigned*)P.ws[rank];
    unsigned* tmo = ctl + W_TMO;
    if (tid == 0) {
        const unsigned t = __hip_atomic_fetch_add(ctl + W_TICKET, 1u, __ATOMIC_RELAXED, __HIP_MEMORY_SCOPE_AGENT);
        *(LAS unsigned*)(lds + STAGE_BYTES) = t / gridDim.x + 1u;
    }
    __syncthreads();
    const unsigned call = (unsigned)__builtin_amdgcn_readfirstlane(*(LAS unsigned*)(lds + STAGE_BYTES));
    __syncthreads();
    const size_t bufoff = CTL_BYTES + (size_t)(call & 1u) * BUF_BYTES;
    const int dest = (rank + xcd) & 7;
    const int pm = dest * 2 + (idx & 1), pn = idx >> 1;
    const int half = idx >> 4;
    const int grp = half ? ((idx - 16) >> 2) : 4;
    const int sh0 = grp < 2 ? 0 : (grp < 4 ? 1 : -1), sh1 = grp < 2 ? 2 : -1;
    const int member = idx & 7;
    auto hook_of = [](int sh) { return sh < 0 ? 8 : (sh == 0 ? 0 : (sh == 1 ? SPLIT - 2 : SPLIT)); };

    auto stage_wait = [&](int h) {
        if (wid == 0) {
            wait_ge(ctl + W_CNT + 32 * h, call * 512u, tmo, call);
            wait_ge(ctl + W_CNT + 32 * ((1 + half) * 8 + h), call * 256u, tmo, call);
            __builtin_amdgcn_fence(__ATOMIC_ACQUIRE, "");
            asm volatile("s_waitcnt vmcnt(0)" ::: "memory");
        }
        asm volatile("" ::: "memory"); __builtin_amdgcn_s_barrier(); asm volatile("" ::: "memory");
    };
    auto do_duty = [&](int shift) {
        int lz = threadIdx.x; asm volatile("" : "+v"(lz)); lz &= 63;
        LAS unsigned* tick = (LAS unsigned*)(lds + STAGE_BYTES + 16);
        if (tid == 0) *tick = 0u;
        if (shift > 0 && wid == 0) wait_ge(ctl + W_DUTY + 32 * ((shift - 1) * 8 + xcd), call * (unsigned)NWG, tmo, call);
        asm volatile("s_waitcnt vmcnt(0) lgkmcnt(0)" ::: "memory"); __builtin_amdgcn_s_barrier(); asm volatile("" ::: "memory");
        const int ubeg = shift == 0 ? 0 : (shift == 1 ? U1 : U2), uend = shift == 0 ? U1 : (shift == 1 ? U2 : U3);
        const int T = (uend - ubeg) / NWG;
        unsigned* dctl = (unsigned*)P.ws[dest];
        char* dbase = P.ws[dest] + bufoff;
        LAS unsigned char* slot0 = lds + wid * 16384;
        auto ticket = [&]() {
            unsigned t = 0;
            if (lz == 0) asm volatile("ds_add_rtn_u32 %0, %1, %2\n\ts_waitcnt lgkmcnt(0)" : "=&v"(t) : "v"((unsigned)(STAGE_BYTES + 16)), "v"(1u) : "memory");
            return (int)__builtin_amdgcn_readfirstlane(t); };
        auto unit_of = [&](int j) { return ubeg + (j >> 3) * (NWG * 8) + member * 8 + (j & 7); };
        int j = wid < PW ? ticket() : T, s_, r, k = 0;
        if (j < T) {
            unsigned* prevc = nullptr;
            unit_decode(unit_of(j), s_, r);
            unit_load(P, s_, dest, r, lz, slot0);
            for (;;) {
                const int jn = ticket(); const bool more = jn < T; int sn, rn;
                unit_decode(unit_of(more ? jn : j), sn, rn);
                unit_load(P, sn, dest, rn, lz, slot0 + ((k + 1) & 1) * 8192);
                if (k == 0) asm volatile("s_waitcnt vmcnt(8)" ::: "memory"); else asm volatile("s_waitcnt vmcnt(16)" ::: "memory");
                unit_store(dbase, s_ * 8 + rank, r, slot0 + (k & 1) * 8192, lz);
                asm volatile("s_waitcnt vmcnt(16) lgkmcnt(0)" ::: "memory");
                if (lz == 0) {
                    if (prevc != nullptr) __hip_atomic_fetch_add(prevc, 1u, __ATOMIC_RELAXED, __HIP_MEMORY_SCOPE_SYSTEM);
                    if (j == T - HANDOFF)
                        __hip_atomic_fetch_add(ctl + W_DUTY + 32 * (shift * 8 + xcd), 1u, __ATOMIC_RELAXED, __HIP_MEMORY_SCOPE_AGENT);
                }
                prevc = dctl + W_CNT + 32 * ((r < 64 ? 0 : (r < 96 ? 1 : 2)) * 8 + s_);
                if (!more) break;
                j = jn; s_ = sn; r = rn; ++k;
            }
            asm volatile("s_waitcnt vmcnt(0)" ::: "memory");
            if (lz == 0) __hip_atomic_fetch_add(prevc, 1u, __ATOMIC_RELAXED, __HIP_MEMORY_SCOPE_SYSTEM);
        }
        asm volatile("" ::: "memory"); __builtin_amdgcn_s_barrier(); asm volatile("" ::: "memory");
        if (shift == 2 && tid == 0) __hip_atomic_fetch_add(ctl + W_DUTY + 32 * (3 * 8 + xcd), 1u, __ATOMIC_RELAXED, __HIP_MEMORY_SCOPE_AGENT);
    };

    const unsigned voff0 = (unsigned)tid * 16u, voff1 = voff0 + 8192u;
    const unsigned ldsw = (unsigned)wid * 1024u;
    const int aoff = wr * 8192 + lane * 16, boff = wc * 4096 + lane * 16;
#define SA(b_, h_) (((b_) * 2 + (h_)) * HTB)
#define SB(b_, h_) ((4 + (b_) * 2 + (h_)) * HTB)
#define STAGE(bufo, gbase) do { \
        __builtin_amdgcn_global_load_lds((const unsigned*)((const char*)(gbase) + voff0), (LAS unsigned*)(lds + (bufo) + ldsw), 16, 0, 0); \
        __builtin_amdgcn_global_load_lds((const unsigned*)((const char*)(gbase) + voff1), (LAS unsigned*)(lds + (bufo) + ldsw + 8192), 16, 0, 0); } while (0)
#define LDA(dst, b_, h_) do { _Pragma("unroll") for (int m = 0; m < 4; ++m) _Pragma("unroll") for (int k = 0; k < 2; ++k) dst[m][k] = *(const LAS bf16x8*)(lds + SA(b_, h_) + aoff + m * 2048 + k * 1024); } while (0)
#define LDB(dst, b_, h_) do { _Pragma("unroll") for (int n = 0; n < 2; ++n) _Pragma("unroll") for (int k = 0; k < 2; ++k) dst[n][k] = *(const LAS bf16x8*)(lds + SB(b_, h_) + boff + n * 2048 + k * 1024); } while (0)
#define MMA(ai, bj, At_, Bt_) do { __builtin_amdgcn_s_setprio(1); _Pragma("unroll") for (int m = 0; m < 4; ++m) _Pragma("unroll") for (int n = 0; n < 2; ++n) _Pragma("unroll") for (int k = 0; k < 2; ++k) \
        acc[ai][bj][m][n] = __builtin_amdgcn_mfma_f32_16x16x32_bf16(Bt_[n][k], At_[m][k], acc[ai][bj][m][n], 0, 0, 0); __builtin_amdgcn_s_setprio(0); } while (0)
#define WAIT_V(n) asm volatile("s_waitcnt vmcnt(" #n ")" ::: "memory")
#define WAIT_L(n) asm volatile("s_waitcnt lgkmcnt(" #n ")" ::: "memory")
#define BAR __builtin_amdgcn_s_barrier()
#define SCHED __builtin_amdgcn_sched_barrier(0)

    f32x4 acc[2][2][4][2];
#pragma unroll
    for (int a = 0; a < 2; ++a)
#pragma unroll
        for (int c = 0; c < 2; ++c)
#pragma unroll
            for (int m = 0; m < 4; ++m)
#pragma unroll
                for (int n = 0; n < 2; ++n) acc[a][c][m][n] = (f32x4){0.f, 0.f, 0.f, 0.f};
    bf16x8 At[4][2], B0[2][2], B1[2][2];
    const char* cA = P.ws[rank] + bufoff + (size_t)pm * 2 * HTB;
    const char* cB = P.ws[rank] + bufoff + OPER_BYTES + (size_t)pn * 2 * HTB;
    constexpr size_t kstep = KT_BYTES, hstep = HTB;

    int kt0 = 0;
    for (int seg = 0; seg < 3; ++seg) {
        const int sh = seg == 0 ? sh0 : (seg == 1 ? sh1 : -1);
        const int kt1 = hook_of(sh) * 8;
        if (kt1 > kt0) {
            stage_wait(kt0 >> 3);
            {
                const char* a0 = cA + (size_t)kt0 * kstep; const char* b0 = cB + (size_t)kt0 * kstep;
                STAGE(SB(0, 0), b0); STAGE(SA(0, 0), a0); STAGE(SB(0, 1), b0 + hstep); STAGE(SA(0, 1), a0 + hstep);
                if (wr == 1) BAR;
                WAIT_V(4); BAR;
                STAGE(SB(1, 0), b0 + kstep); STAGE(SA(1, 0), a0 + kstep); STAGE(SB(1, 1), b0 + hstep + kstep);
                WAIT_V(6); BAR;
            }
            for (int t = kt0; t < kt1; t += 2) {
                const bool last = (t == kt1 - 2);
                const char* a1 = cA + (size_t)(t + 1) * kstep;
                const char* a2 = last ? cA : cA + (size_t)(t + 2) * kstep; const char* b2 = last ? cB : cB + (size_t)(t + 2) * kstep;
                const char* a3 = a2 + kstep; const char* b3 = b2 + kstep;
                if (((t + 2) & 7) == 0 && !last) stage_wait((t + 2) >> 3);
                LDB(B0, 0, 0); SCHED; LDA(At, 0, 0); STAGE(SA(1, 1), a1 + hstep);
                WAIT_L(8); BAR; WAIT_L(0); MMA(0, 0, At, B0); BAR; SCHED;
                LDB(B1, 0, 1); STAGE(SB(0, 0), b2);
                BAR; WAIT_L(0); MMA(0, 1, At, B1); BAR;
                LDA(At, 0, 1); STAGE(SA(0, 0), a2);
                BAR; WAIT_L(0); MMA(1, 0, At, B0); BAR; SCHED;
                STAGE(SB(0, 1), b2 + hstep);
                WAIT_V(6); BAR; MMA(1, 1, At, B1); BAR;
                LDB(B0, 1, 0); SCHED; LDA(At, 1, 0); STAGE(SA(0, 1), a2 + hstep);
                WAIT_L(8); BAR; WAIT_L(0); MMA(0, 0, At, B0); BAR; SCHED;
                LDB(B1, 1, 1); STAGE(SB(1, 0), b3);
                BAR; WAIT_L(0); MMA(0, 1, At, B1); BAR;
                LDA(At, 1, 1); STAGE(SA(1, 0), a3);
                BAR; WAIT_L(0); MMA(1, 0, At, B0); BAR; SCHED;
                STAGE(SB(1, 1), b3 + hstep);
                WAIT_V(6); BAR; MMA(1, 1, At, B1); BAR;
            }
            WAIT_V(0);
            if (wr == 0) BAR;
            BAR;
        }
        if (sh >= 0) do_duty(sh);
        kt0 = kt1;
    }
    if (half == 0) {
        if (wid == 0) wait_ge(ctl + W_DUTY + 32 * (3 * 8 + xcd), call * (unsigned)NWG, tmo, call);
        asm volatile("" ::: "memory"); __builtin_amdgcn_s_barrier(); asm volatile("" ::: "memory");
    }
    {
        const bool bad = (unsigned)__builtin_amdgcn_readfirstlane(__hip_atomic_load(tmo, __ATOMIC_RELAXED, __HIP_MEMORY_SCOPE_AGENT)) == call;
        char* obase = P.out[dest] + ((size_t)((pm & 1) * 256 + wr * 64 + fr) * NTOT + (size_t)rank * NSL + pn * 256 + wc * 32 + 8 * fq) * 2;
#pragma unroll
        for (int ai = 0; ai < 2; ++ai)
#pragma unroll
            for (int m = 0; m < 4; ++m) {
                char* rowp = obase + (size_t)(ai * 128 + m * 16) * NTOT * 2;
#pragma unroll
                for (int bj = 0; bj < 2; ++bj) {
                    const f32x4 v0 = acc[ai][bj][m][0], v1 = acc[ai][bj][m][1];
                    u32x4 o;
                    o.x = cvt_pk_bf16(gelu1(v0[0]), gelu1(v0[1])); o.y = cvt_pk_bf16(gelu1(v0[2]), gelu1(v0[3]));
                    o.z = cvt_pk_bf16(gelu1(v1[0]), gelu1(v1[1])); o.w = cvt_pk_bf16(gelu1(v1[2]), gelu1(v1[3]));
                    if (bad) o = (u32x4){0x7fc07fc0u, 0x7fc07fc0u, 0x7fc07fc0u, 0x7fc07fc0u};
                    st_sys(rowp + bj * 256, o);
                }
            }
    }
    asm volatile("s_waitcnt vmcnt(0)" ::: "memory");
    __syncthreads();
    if (tid == 0) {
        __hip_atomic_fetch_add((unsigned*)P.ws[dest] + W_OUT, 1u, __ATOMIC_RELAXED, __HIP_MEMORY_SCOPE_SYSTEM);
    }
    if (b == 0 && wid == 0) {
        wait_ge(ctl + W_OUT, call * 256u, tmo, call);
        __builtin_amdgcn_fence(__ATOMIC_ACQUIRE, "");
        if ((unsigned)__builtin_amdgcn_readfirstlane(__hip_atomic_load(tmo, __ATOMIC_RELAXED, __HIP_MEMORY_SCOPE_AGENT)) == call && lane == 0)
            *(unsigned short*)P.out[rank] = (unsigned short)0x7fc0u;
    }
}
}

extern "C" void kernel_launch(int rank, int nranks, ncclComm_t comm,
                              hipStream_t stream,
                              void* const* d_in, const int* in_sizes, int n_in,
                              void* d_out, int out_size,
                              const int* in_dtypes, int out_dtype,
                              void* const* d_out_all,
                              void* const* d_ws_all, size_t ws_bytes) {
    if (nranks != NR || ws_bytes < WS_NEED || n_in < 2) { fprintf(stderr, "unexpected configuration\n"); return; }
    static bool attr = false;
    if (!attr) { HIP_CHECK(hipFuncSetAttribute((const void*)fused_gemm, hipFuncAttributeMaxDynamicSharedMemorySize, LDS_BYTES)); attr = true; }
    Params P;
    P.x = (const bf16_t*)d_in[0]; P.w = (const bf16_t*)d_in[1]; P.rank = rank;
    for (int p = 0; p < NR; ++p) { P.ws[p] = (char*)d_ws_all[p]; P.out[p] = (char*)d_out_all[p]; }
    hipLaunchKernelGGL(fused_gemm, dim3(256), dim3(512), LDS_BYTES, stream, P);
    HIP_CHECK(hipGetLastError());
}
```
